```python
import numpy as np

kernel_name = "gpu_gemm_ar_m4096_k4096_n16384_w8a8_dqsilu"


ABSMAX_RTOL = 0.001

COMPUTE_DTYPE = "int8"
EPILOGUE = "dequant_silu"
IN_QUANT = "int8"
OUT_QUANT = None
IN_DTYPES = {"x": "int8", "w_mat": "int8",
             "scale_x": "float32", "scale_w": "float32"}
WORLD_SIZE = 8

def setup_inputs():

    x = np.random.randint(-127, 128, size=(4096, 4096)).astype(np.int8)
    w_mat = np.random.randint(-127, 128, size=(4096, 16384)).astype(np.int8)
    scale_x = (0.02 * (1.0 + np.random.rand(1))).astype(np.float32)
    scale_w = (0.02 * (1.0 + np.random.rand(1))).astype(np.float32)
    return {"x": x, "w_mat": w_mat, "scale_x": scale_x, "scale_w": scale_w}

def reference(x, w_mat, scale_x, scale_w):
    acc = x.astype(np.float64) @ w_mat.astype(np.float64)
    y = (acc.astype(np.float32) * (scale_x * scale_w)).astype(np.float32)
    return (y / (1.0 + np.exp(-np.clip(y, -60.0, 60.0)))).astype(np.float32)
```

```cpp
#include <hip/hip_runtime.h>
#include <rccl/rccl.h>
#include <cstdio>
#include <cstdint>

#define HIP_CHECK(expr) do { hipError_t _e = (expr); if (_e != hipSuccess) { \
    fprintf(stderr, "HIP error %s at %s:%d\n", hipGetErrorString(_e), __FILE__, __LINE__); \
    return; } } while (0)

namespace {

#define LAS __attribute__((address_space(3)))
typedef int i32x4 __attribute__((ext_vector_type(4)));
typedef unsigned u32x4 __attribute__((ext_vector_type(4)));
typedef float f32x4 __attribute__((ext_vector_type(4)));

constexpr int GM = 4096, GK = 4096, GN = 16384, NRANK = 8, KSL = GK / NRANK;
constexpr int MT = GM / 256, NT = GN / 256;
constexpr unsigned KTB = 32768;
constexpr unsigned CHUNKB = 4 * KTB;
constexpr int NCHUNK = MT + NT;
constexpr int UNITS_PER_CHUNK = 32;
constexpr int NUNITS = NCHUNK * UNITS_PER_CHUNK;

constexpr size_t WS_CTR = 0;
constexpr size_t WS_TMO = 256;
constexpr size_t WS_FLAGS = 4096;
constexpr size_t WS_DATA = 1u << 20;
constexpr size_t XA_BYTES = (size_t)NRANK * MT * CHUNKB;
constexpr size_t WB_BYTES = (size_t)NRANK * NT * CHUNKB;
constexpr size_t PAR_BYTES = XA_BYTES + WB_BYTES;
constexpr size_t WS_NEED = WS_DATA + 2 * PAR_BYTES;

constexpr int NPUSH = 16;
constexpr int NBLK = 256;
constexpr int NGEMM = NBLK - NPUSH;
constexpr int NTILES = MT * NT;

constexpr unsigned HTB = 16384;
constexpr unsigned STAGE_BYTES = 8 * HTB;
constexpr unsigned LDS_FLAG = STAGE_BYTES;
constexpr unsigned LDS_MISC = STAGE_BYTES + 256;
constexpr unsigned LDS_BYTES = STAGE_BYTES + 512;

constexpr unsigned long long DEADLINE_TICKS = 50000000ull;

struct Params {
    const signed char* x;
    const signed char* w;
    const float* sx;
    const float* sw;
    float* out;
    char* const* ws_all;
    int rank;
};

__device__ __forceinline__ unsigned perm(unsigned hi, unsigned lo, unsigned sel) { return __builtin_amdgcn_perm(hi, lo, sel); }

__device__ __forceinline__ void pusher(const Params& p, unsigned call, int bid) {
    const unsigned tid = threadIdx.x, lane = tid & 63;
    const unsigned wid = __builtin_amdgcn_readfirstlane(tid >> 6);
    const unsigned gw = (unsigned)bid * 8 + wid, nw = NPUSH * 8;
    const size_t par = WS_DATA + (size_t)(call & 1) * PAR_BYTES;
    __amdgpu_buffer_rsrc_t rs[NRANK];
#pragma unroll
    for (int i = 0; i < NRANK; ++i) {
        const unsigned d = (unsigned)(p.rank + 1 + i) & 7u;
        rs[i] = __builtin_amdgcn_make_buffer_rsrc(p.ws_all[d] + par, 0, (unsigned)PAR_BYTES, 0x00020000);
    }
    unsigned* myflag = (unsigned*)(p.ws_all[(unsigned)(p.rank + 1 + (lane & 7)) & 7u] + WS_FLAGS + (size_t)p.rank * NCHUNK * 128);
    const unsigned g = lane >> 4, r = lane & 15;
    for (unsigned u = gw; u < (unsigned)NUNITS; u += nw) {
        const unsigned q = u >> 5, s = u & 31;
        u32x4 v[4];
        unsigned doff[4], chunk;
        if (q < 32 && !(q & 1)) {
            const unsigned mt = q >> 1, mf = s >> 1, kp = s & 1;
            chunk = mt;
            const signed char* src = p.x + (size_t)(mt * 256 + mf * 16 + r) * KSL + kp * 256 + g * 16;
#pragma unroll
            for (int j = 0; j < 4; ++j) {
                v[j] = *(const u32x4*)(src + j * 64);
                doff[j] = ((unsigned)p.rank * MT + mt) * CHUNKB + (kp * 2 + (j >> 1)) * KTB + mf * 2048 + (j & 1) * 1024 + lane * 16;
            }
        } else {
            const unsigned nt = q < 32 ? (q >> 1) : (q - 16), wc = s & 3, kt = (s >> 2) & 3, kb = s >> 4;
            chunk = MT + nt;
            const signed char* src = p.w + (size_t)(kt * 128 + kb * 64 + g * 16) * GN + nt * 256 + wc * 64 + r * 4;
            unsigned a[16];
#pragma unroll
            for (int j = 0; j < 16; ++j) a[j] = *(const unsigned*)(src + (size_t)j * GN);
#pragma unroll
            for (int b = 0; b < 4; ++b) {
                const unsigned t0 = perm(a[4 * b + 1], a[4 * b + 0], 0x05010400u), t1 = perm(a[4 * b + 1], a[4 * b + 0], 0x07030602u);
                const unsigned t2 = perm(a[4 * b + 3], a[4 * b + 2], 0x05010400u), t3 = perm(a[4 * b + 3], a[4 * b + 2], 0x07030602u);
                v[0][b] = perm(t2, t0, 0x05040100u); v[1][b] = perm(t2, t0, 0x07060302u);
                v[2][b] = perm(t3, t1, 0x05040100u); v[3][b] = perm(t3, t1, 0x07060302u);
            }
#pragma unroll
            for (int i = 0; i < 4; ++i)
                doff[i] = (unsigned)XA_BYTES + ((unsigned)p.rank * NT + nt) * CHUNKB + kt * KTB + (i >> 1) * HTB + (wc * 2 + (i & 1)) * 2048 + kb * 1024 + lane * 16;
        }
#pragma unroll
        for (int i = 0; i < NRANK; ++i)
#pragma unroll
            for (int j = 0; j < 4; ++j) __builtin_amdgcn_raw_buffer_store_b128(v[j], rs[i], doff[j], 0, 17  );
        asm volatile("s_waitcnt vmcnt(0)" ::: "memory");
        if (lane < NRANK) __hip_atomic_fetch_add(myflag + chunk * 32, 1u, __ATOMIC_RELAXED, __HIP_MEMORY_SCOPE_SYSTEM);
    }
}

__device__ __forceinline__ void tile_of(int idx, int& mt, int& nt) {
    if (idx < 256) {
        int s = (int)__builtin_sqrtf((float)idx + 0.5f);
        const int r = idx - s * s;
        if (r <= s) { mt = s; nt = r; } else { mt = r - s - 1; nt = s; }
    } else {
        nt = 16 + ((idx - 256) >> 4); mt = (idx - 256) & 15;
    }
}

__global__ __launch_bounds__(512, 2) void fused_kernel(Params p) {
    extern __shared__ __attribute__((aligned(16))) unsigned char lds_raw[];
    LAS unsigned char* lds = (LAS unsigned char*)lds_raw;
    const int tid = threadIdx.x, lane = tid & 63;
    const int wid = __builtin_amdgcn_readfirstlane(tid >> 6);
    char* myws = p.ws_all[p.rank];

    if (tid == 0) {
        const unsigned ticket = __hip_atomic_fetch_add((unsigned*)(myws + WS_CTR), 1u, __ATOMIC_RELAXED, __HIP_MEMORY_SCOPE_AGENT);
        *(LAS unsigned*)(lds + LDS_MISC) = ticket / gridDim.x + 1u;
        __builtin_amdgcn_fence(__ATOMIC_ACQUIRE, "");
        asm volatile("s_waitcnt vmcnt(0) lgkmcnt(0)" ::: "memory");
    }
    __syncthreads();
    const unsigned call = __builtin_amdgcn_readfirstlane(*(LAS unsigned*)(lds + LDS_MISC));

    if ((int)blockIdx.x < NPUSH) { pusher(p, call, blockIdx.x); return; }

    const int wr = wid >> 2, wc = wid & 3;
    const unsigned target = call * UNITS_PER_CHUNK;
    const char* XA = myws + WS_DATA + (size_t)(call & 1) * PAR_BYTES;
    const char* WB = XA + XA_BYTES;
    const unsigned* flags = (const unsigned*)(myws + WS_FLAGS);
    unsigned* tmo = (unsigned*)(myws + WS_TMO);
    const float scale = p.sx[0] * p.sw[0];
    bool bad = false;

    const unsigned voff = (unsigned)tid * 16u;
    const unsigned ldsw = (unsigned)wid * 1024u;
    const unsigned aoff = (unsigned)wr * 8192u + (unsigned)lane * 16u, boff = (unsigned)wc * 4096u + (unsigned)lane * 16u;

#define SA(b, h) (((b) * 2 + (h)) * HTB)
#define SB(b, h) ((4 + (b) * 2 + (h)) * HTB)
#define STAGE(bufoff, gbase) do { _Pragma("unroll") for (int _i = 0; _i < 2; ++_i) \
        __builtin_amdgcn_global_load_lds((const unsigned*)((const char*)(gbase) + voff + _i * 8192), (LAS unsigned*)(lds + (bufoff) + ldsw + _i * 8192), 16, 0, 0); } while (0)
#define LDA(dst, b, h) do { _Pragma("unroll") for (int m = 0; m < 4; ++m) _Pragma("unroll") for (int k = 0; k < 2; ++k) dst[m][k] = *(const LAS i32x4*)(lds + SA(b, h) + aoff + m * 2048 + k * 1024); } while (0)
#define LDB(dst, b, h) do { _Pragma("unroll") for (int n = 0; n < 2; ++n) _Pragma("unroll") for (int k = 0; k < 2; ++k) dst[n][k] = *(const LAS i32x4*)(lds + SB(b, h) + boff + n * 2048 + k * 1024); } while (0)
#define MMA(ai, bj, At, Bt) do { __builtin_amdgcn_s_setprio(1); _Pragma("unroll") for (int m = 0; m < 4; ++m) _Pragma("unroll") for (int n = 0; n < 2; ++n) _Pragma("unroll") for (int k = 0; k < 2; ++k) \
        acc[ai][bj][m][n] = __builtin_amdgcn_mfma_i32_16x16x64_i8(At[m][k], Bt[n][k], acc[ai][bj][m][n], 0, 0, 0); __builtin_amdgcn_s_setprio(0); } while (0)
#define WAIT_V(n) asm volatile("s_waitcnt vmcnt(" #n ")" ::: "memory")
#define WAIT_L(n) asm volatile("s_waitcnt lgkmcnt(" #n ")" ::: "memory")
#define BAR __builtin_amdgcn_s_barrier()
#define SCHED __builtin_amdgcn_sched_barrier(0)

    auto wait_slice = [&](int src, int mt, int nt) {
        const unsigned* f = flags + ((size_t)src * NCHUNK + ((lane & 1) ? (MT + nt) : mt)) * 32;
        const unsigned long long t0 = __builtin_amdgcn_s_memrealtime();
        for (;;) {
            const unsigned v = __hip_atomic_load(f, __ATOMIC_RELAXED, __HIP_MEMORY_SCOPE_SYSTEM);
            if (__all((int)(v - target) >= 0)) break;
            if (__hip_atomic_load(tmo, __ATOMIC_RELAXED, __HIP_MEMORY_SCOPE_AGENT) == call || __builtin_amdgcn_s_memrealtime() - t0 > DEADLINE_TICKS) {
                __hip_atomic_store(tmo, call, __ATOMIC_RELAXED, __HIP_MEMORY_SCOPE_AGENT);
                bad = true;
                break;
            }
            __builtin_amdgcn_s_sleep(8);
        }
        asm volatile("s_waitcnt vmcnt(0)" ::: "memory");
    };

    int tidx = (int)blockIdx.x - NPUSH;
    int mt, nt;
    tile_of(tidx, mt, nt);
    i32x4 acc[2][2][4][2];
#pragma unroll
    for (int a = 0; a < 2; ++a)
#pragma unroll
        for (int b = 0; b < 2; ++b)
#pragma unroll
            for (int m = 0; m < 4; ++m)
#pragma unroll
                for (int n = 0; n < 2; ++n) acc[a][b][m][n] = (i32x4){0, 0, 0, 0};
    i32x4 At[4][2], B0[2][2], B1[2][2];

    const char* cA = XA + ((size_t)p.rank * MT + mt) * CHUNKB;
    const char* cB = WB + ((size_t)p.rank * NT + nt) * CHUNKB;
    wait_slice(p.rank, mt, nt);
    STAGE(SB(0, 0), cB); STAGE(SA(0, 0), cA); STAGE(SB(0, 1), cB + HTB); STAGE(SA(0, 1), cA + HTB);
    if (wr == 1) BAR;
    WAIT_V(4); BAR;
    STAGE(SB(1, 0), cB + KTB); STAGE(SA(1, 0), cA + KTB); STAGE(SB(1, 1), cB + KTB + HTB);
    WAIT_V(6); BAR;
    for (;;) {
        const int nidx = tidx + NGEMM;
        const bool has_next = nidx < NTILES;
        int nmt = mt, nnt = nt;
        if (has_next) tile_of(nidx, nmt, nnt);
        for (int t = 0; t < 32; t += 2) {
            const char* a1 = cA + KTB;
            const char *a2, *b2;
            if ((t & 2) == 0) { a2 = cA + 2 * KTB; b2 = cB + 2 * KTB; }
            else {
                const int sl = (t >> 2) + 1;
                const int src = (p.rank + sl) & 7;
                const int m2 = sl == 8 ? nmt : mt, n2 = sl == 8 ? nnt : nt;
                a2 = XA + ((size_t)src * MT + m2) * CHUNKB;
                b2 = WB + ((size_t)src * NT + n2) * CHUNKB;
                if (sl < 8 || has_next) wait_slice(src, m2, n2);
            }
            const char* a3 = a2 + KTB; const char* b3 = b2 + KTB;
            LDB(B0, 0, 0); SCHED; LDA(At, 0, 0); STAGE(SA(1, 1), a1 + HTB);
            WAIT_L(8); BAR; WAIT_L(0); MMA(0, 0, At, B0); BAR; SCHED;
            LDB(B1, 0, 1); STAGE(SB(0, 0), b2);
            BAR; WAIT_L(0); MMA(0, 1, At, B1); BAR;
            LDA(At, 0, 1); STAGE(SA(0, 0), a2);
            BAR; WAIT_L(0); MMA(1, 0, At, B0); BAR; SCHED;
            STAGE(SB(0, 1), b2 + HTB);
            WAIT_V(6); BAR; MMA(1, 1, At, B1); BAR;
            LDB(B0, 1, 0); SCHED; LDA(At, 1, 0); STAGE(SA(0, 1), a2 + HTB);
            WAIT_L(8); BAR; WAIT_L(0); MMA(0, 0, At, B0); BAR; SCHED;
            LDB(B1, 1, 1); STAGE(SB(1, 0), b3);
            BAR; WAIT_L(0); MMA(0, 1, At, B1); BAR;
            LDA(At, 1, 1); STAGE(SA(1, 0), a3);
            BAR; WAIT_L(0); MMA(1, 0, At, B0); BAR; SCHED;
            STAGE(SB(1, 1), b3 + HTB);
            WAIT_V(6); BAR; MMA(1, 1, At, B1); BAR;
            cA = a2; cB = b2;
        }
        {
            const int fr = lane & 15, fq = lane >> 4;
            float* obase = p.out + (size_t)(mt * 256 + wr * 64 + fq * 4) * GN + nt * 256 + wc * 64 + fr * 4;
            const float qnan = __builtin_nanf("");
#pragma unroll
            for (int ai = 0; ai < 2; ++ai)
#pragma unroll
                for (int m = 0; m < 4; ++m)
#pragma unroll
                    for (int j = 0; j < 4; ++j) {
                        f32x4 o;
#pragma unroll
                        for (int i = 0; i < 4; ++i) {
                            const float y = (float)acc[ai][i >> 1][m][i & 1][j] * scale;
                            const float c = __builtin_amdgcn_fmed3f(y, -60.0f, 60.0f);
                            const float e = __builtin_amdgcn_exp2f(c * -1.4426950408889634f);
                            o[i] = y * __builtin_amdgcn_rcpf(1.0f + e);
                        }
                        if (bad) o = (f32x4){qnan, qnan, qnan, qnan};
                        *(f32x4*)(obase + (size_t)(ai * 128 + m * 16 + j) * GN) = o;
                    }
        }
        if (!has_next) break;
#pragma unroll
        for (int a = 0; a < 2; ++a)
#pragma unroll
            for (int b = 0; b < 2; ++b)
#pragma unroll
                for (int m = 0; m < 4; ++m)
#pragma unroll
                    for (int n = 0; n < 2; ++n) acc[a][b][m][n] = (i32x4){0, 0, 0, 0};
        tidx = nidx; mt = nmt; nt = nnt;
    }
    WAIT_V(0);
    if (wr == 0) BAR;
#undef SA
#undef SB
#undef STAGE
#undef LDA
#undef LDB
#undef MMA
#undef WAIT_V
#undef WAIT_L
#undef BAR
#undef SCHED
}

}

extern "C" void kernel_launch(int rank, int nranks, ncclComm_t comm,
                              hipStream_t stream,
                              void* const* d_in, const int* in_sizes, int n_in,
                              void* d_out, int out_size,
                              const int* in_dtypes, int out_dtype,
                              void* const* d_out_all,
                              void* const* d_ws_all, size_t ws_bytes) {
    (void)comm; (void)in_dtypes; (void)out_dtype; (void)d_out_all;
    if (nranks != NRANK || rank < 0 || rank >= NRANK || n_in < 4 || ws_bytes < WS_NEED ||
        in_sizes[0] != GM * KSL || in_sizes[1] != KSL * GN || out_size != GM * GN) {
        fprintf(stderr, "kernel_launch: unexpected problem shape\n");
        return;
    }
    static bool attr_set = false;
    if (!attr_set) {
        HIP_CHECK(hipFuncSetAttribute((const void*)fused_kernel, hipFuncAttributeMaxDynamicSharedMemorySize, LDS_BYTES));
        attr_set = true;
    }
    Params p;
    p.x = (const signed char*)d_in[0];
    p.w = (const signed char*)d_in[1];
    p.sx = (const float*)d_in[2];
    p.sw = (const float*)d_in[3];
    p.out = (float*)d_out;
    p.ws_all = (char* const*)d_ws_all;
    p.rank = rank;
    fused_kernel<<<NBLK, 512, LDS_BYTES, stream>>>(p);
    HIP_CHECK(hipGetLastError());
}
```

```cpp
#include <hip/hip_runtime.h>
#include <rccl/rccl.h>
#include <cstdio>
#include <cstdint>

#define HIP_CHECK(expr) do { hipError_t _e = (expr); if (_e != hipSuccess) { \
    fprintf(stderr, "HIP error %s at %s:%d\n", hipGetErrorString(_e), __FILE__, __LINE__); \
    return; } } while (0)

namespace {

#define LAS __attribute__((address_space(3)))
#define GAS __attribute__((address_space(1)))
typedef int i32x4 __attribute__((ext_vector_type(4)));
typedef unsigned u32x4 __attribute__((ext_vector_type(4)));
typedef float f32x4 __attribute__((ext_vector_type(4)));

constexpr int GM = 4096, GK = 4096, GN = 16384, NRANK = 8, KSL = GK / NRANK;
constexpr int MT = GM / 256, NT = GN / 256;
constexpr unsigned KTB = 32768;
constexpr unsigned CHUNKB = 4 * KTB;
constexpr int NCHUNK = MT + NT;
constexpr int UNITS_PER_CHUNK = 32;
constexpr int NUNITS = NCHUNK * UNITS_PER_CHUNK;

constexpr size_t WS_CTR = 0;
constexpr size_t WS_TMO = 256;
constexpr size_t WS_QTICK = 1024;
constexpr size_t WS_FLAGS = 4096;
constexpr size_t WS_DATA = 1u << 20;
constexpr size_t XA_BYTES = (size_t)NRANK * MT * CHUNKB;
constexpr size_t WB_BYTES = (size_t)NRANK * NT * CHUNKB;
constexpr size_t PAR_BYTES = XA_BYTES + WB_BYTES;
constexpr size_t WS_NEED = WS_DATA + 2 * PAR_BYTES;

#define STAMP(i) do {} while (0)
constexpr int NPUSH = 16;
constexpr int NBLK = 256;
constexpr int NGEMM = NBLK - NPUSH;
constexpr int NTILES = MT * NT;
constexpr int WPG = NGEMM / 8;
constexpr int NFULL = 4 * WPG;
constexpr int NJOB = NFULL + 32;

constexpr unsigned HTB = 16384;
constexpr unsigned STAGE_BYTES = 8 * HTB;
constexpr unsigned LDS_FLAG = STAGE_BYTES;
constexpr unsigned LDS_MISC = STAGE_BYTES + 256;
constexpr unsigned LDS_BYTES = STAGE_BYTES + 512;

#define POLL_SLEEP 16
constexpr unsigned long long DEADLINE_TICKS = 50000000ull;

struct Params {
    const signed char* x;
    const signed char* w;
    const float* sx;
    const float* sw;
    float* out;
    char* const* ws_all;
    int rank;
    unsigned kdef;
    unsigned short order[8 * NJOB];
    unsigned korder[256];
};
constexpr int ROT_X = 2, ROT_W = 10;

__device__ __forceinline__ unsigned perm(unsigned hi, unsigned lo, unsigned sel) { return __builtin_amdgcn_perm(hi, lo, sel); }

__device__ __forceinline__ char* uniform_ptr(char* q) {
    const unsigned long long v = (unsigned long long)q;
    const unsigned lo = __builtin_amdgcn_readfirstlane((unsigned)v), hi = __builtin_amdgcn_readfirstlane((unsigned)(v >> 32));
    return (char*)(((unsigned long long)hi << 32) | lo);
}

struct PushUnit { unsigned a[16]; };

__device__ __forceinline__ void push_load(const Params& p, unsigned u, unsigned g, unsigned r, PushUnit& d) {
    const unsigned q = u >> 5, s = u & 31;
    if (q < 32 && !(q & 1)) {
        const unsigned mt = ((q >> 1) + ROT_X * p.rank) & 15, mf = s >> 1, kp = s & 1;
        const signed char* src = p.x + (size_t)(mt * 256 + mf * 16 + r) * KSL + kp * 256 + g * 16;
#pragma unroll
        for (int j = 0; j < 4; ++j) { const u32x4 t = *(const u32x4*)(src + j * 64); d.a[4 * j] = t[0]; d.a[4 * j + 1] = t[1]; d.a[4 * j + 2] = t[2]; d.a[4 * j + 3] = t[3]; }
    } else {
        const unsigned nt = q < 32 ? (((q >> 1) + ROT_W * p.rank) & 15) : (q - 16), wc = s & 3, kt = (s >> 2) & 3, kb = s >> 4;
        const signed char* src = p.w + (size_t)(kt * 128 + kb * 64 + g * 16) * GN + nt * 256 + wc * 64 + r * 4;
#pragma unroll
        for (int j = 0; j < 16; ++j) d.a[j] = *(const unsigned*)(src + (size_t)j * GN);
    }
}

__device__ __forceinline__ void pusher(const Params& p, unsigned call, int bid) {
    const unsigned tid = threadIdx.x, lane = tid & 63;
    const unsigned wid = __builtin_amdgcn_readfirstlane(tid >> 6);
    const unsigned gw = (unsigned)bid * 8 + wid, nw = NPUSH * 8;
    const size_t par = WS_DATA + (size_t)(call & 1) * PAR_BYTES;
    __amdgpu_buffer_rsrc_t rs[NRANK];
#pragma unroll
    for (int i = 0; i < NRANK; ++i) {
        const unsigned d = (unsigned)(p.rank + 1 + i) & 7u;
        rs[i] = __builtin_amdgcn_make_buffer_rsrc(uniform_ptr(p.ws_all[d]) + par, 0, (unsigned)PAR_BYTES, 0x00020000);
    }
    GAS unsigned* myflag = (GAS unsigned*)(p.ws_all[(unsigned)(p.rank + 1 + (lane & 7)) & 7u] + WS_FLAGS + (size_t)p.rank * NCHUNK * 128);
    const unsigned g = lane >> 4, r = lane & 15;
    PushUnit cur, nxt;
    push_load(p, gw, g, r, cur);
#pragma unroll
    for (int j = 0; j < 16; ++j) asm volatile("" :: "v"(cur.a[j]));
    unsigned prev_chunk = 0xffffffffu;
    for (unsigned u = gw; u < (unsigned)NUNITS; u += nw) {
        const unsigned q = u >> 5, s = u & 31;
        u32x4 v[4];
        unsigned doff[4], chunk;
        if (q < 32 && !(q & 1)) {
            const unsigned mt = ((q >> 1) + ROT_X * p.rank) & 15, mf = s >> 1, kp = s & 1;
            chunk = mt;
#pragma unroll
            for (int j = 0; j < 4; ++j) {
                v[j] = (u32x4){cur.a[4 * j], cur.a[4 * j + 1], cur.a[4 * j + 2], cur.a[4 * j + 3]};
                doff[j] = ((unsigned)p.rank * MT + mt) * CHUNKB + (kp * 2 + (j >> 1)) * KTB + mf * 2048 + (j & 1) * 1024 + lane * 16;
            }
        } else {
            const unsigned nt = q < 32 ? (((q >> 1) + ROT_W * p.rank) & 15) : (q - 16), wc = s & 3, kt = (s >> 2) & 3, kb = s >> 4;
            chunk = MT + nt;
#pragma unroll
            for (int b = 0; b < 4; ++b) {
                const unsigned t0 = perm(cur.a[4 * b + 1], cur.a[4 * b + 0], 0x05010400u), t1 = perm(cur.a[4 * b + 1], cur.a[4 * b + 0], 0x07030602u);
                const unsigned t2 = perm(cur.a[4 * b + 3], cur.a[4 * b + 2], 0x05010400u), t3 = perm(cur.a[4 * b + 3], cur.a[4 * b + 2], 0x07030602u);
                v[0][b] = perm(t2, t0, 0x05040100u); v[1][b] = perm(t2, t0, 0x07060302u);
                v[2][b] = perm(t3, t1, 0x05040100u); v[3][b] = perm(t3, t1, 0x07060302u);
            }
#pragma unroll
            for (int i = 0; i < 4; ++i)
                doff[i] = (unsigned)XA_BYTES + ((unsigned)p.rank * NT + nt) * CHUNKB + kt * KTB + (i >> 1) * HTB + (wc * 2 + (i & 1)) * 2048 + kb * 1024 + lane * 16;
        }
        if (u + nw < (unsigned)NUNITS) push_load(p, u + nw, g, r, nxt);
#pragma unroll
        for (int i = 0; i < NRANK; ++i)
#pragma unroll
            for (int j = 0; j < 4; ++j) __builtin_amdgcn_raw_buffer_store_b128(v[j], rs[i], doff[j], 0, 17  );
        asm volatile("s_waitcnt vmcnt(32)" ::: "memory");
        if (prev_chunk != 0xffffffffu && lane < NRANK) __hip_atomic_fetch_add(myflag + prev_chunk * 32, 1u, __ATOMIC_RELAXED, __HIP_MEMORY_SCOPE_SYSTEM);
        STAMP(1 + (u - gw) / nw);
        __builtin_amdgcn_s_barrier();
        prev_chunk = chunk;
        cur = nxt;
    }
    asm volatile("s_waitcnt vmcnt(0)" ::: "memory");
    if (lane < NRANK) __hip_atomic_fetch_add(myflag + prev_chunk * 32, 1u, __ATOMIC_RELAXED, __HIP_MEMORY_SCOPE_SYSTEM);
}

__global__ __launch_bounds__(512, 2) void fused_kernel(Params p) {
    extern __shared__ __attribute__((aligned(16))) unsigned char lds_raw[];
    LAS unsigned char* lds = (LAS unsigned char*)lds_raw;
    const int tid = threadIdx.x, lane = tid & 63;
    const int wid = __builtin_amdgcn_readfirstlane(tid >> 6);
    char* myws = uniform_ptr(p.ws_all[p.rank]);

    if (tid == 0) {
        const unsigned ticket = __hip_atomic_fetch_add((unsigned*)(myws + WS_CTR), 1u, __ATOMIC_RELAXED, __HIP_MEMORY_SCOPE_AGENT);
        *(LAS unsigned*)(lds + LDS_MISC) = ticket / gridDim.x + 1u;
        *(LAS unsigned*)(lds + LDS_MISC + 4) = 0u;
        __builtin_amdgcn_fence(__ATOMIC_ACQUIRE, "");
        asm volatile("s_waitcnt vmcnt(0) lgkmcnt(0)" ::: "memory");
    }
    STAMP(0);
    __syncthreads();
    const unsigned call = __builtin_amdgcn_readfirstlane(*(LAS unsigned*)(lds + LDS_MISC));

    if ((int)blockIdx.x < NPUSH) pusher(p, call, blockIdx.x);

    const int wr = wid >> 2, wc = wid & 3;
    const unsigned target = call * UNITS_PER_CHUNK;
    const char* XA = myws + WS_DATA + (size_t)(call & 1) * PAR_BYTES;
    const char* WB = XA + XA_BYTES;
    const unsigned* flags = (const unsigned*)(myws + WS_FLAGS);
    GAS unsigned* tmo = (GAS unsigned*)(myws + WS_TMO);
    const float scale = p.sx[0] * p.sw[0];

    const unsigned voff = (unsigned)tid * 16u;
    const unsigned ldsw = (unsigned)wid * 1024u;
    const unsigned aoff = (unsigned)wr * 8192u + (unsigned)lane * 16u, boff = (unsigned)wc * 4096u + (unsigned)lane * 16u;

#define SA(b, h) (((b) * 2 + (h)) * HTB)
#define SB(b, h) ((4 + (b) * 2 + (h)) * HTB)
#define STAGE(bufoff, gbase) do { _Pragma("unroll") for (int _i = 0; _i < 2; ++_i) \
        __builtin_amdgcn_global_load_lds((const unsigned*)((const char*)(gbase) + voff + _i * 8192), (LAS unsigned*)(lds + (bufoff) + ldsw + _i * 8192), 16, 0, 0); } while (0)
#define LDA(dst, b, h) do { _Pragma("unroll") for (int m = 0; m < 4; ++m) _Pragma("unroll") for (int k = 0; k < 2; ++k) dst[m][k] = *(const LAS i32x4*)(lds + SA(b, h) + aoff + m * 2048 + k * 1024); } while (0)
#define LDB(dst, b, h) do { _Pragma("unroll") for (int n = 0; n < 2; ++n) _Pragma("unroll") for (int k = 0; k < 2; ++k) dst[n][k] = *(const LAS i32x4*)(lds + SB(b, h) + boff + n * 2048 + k * 1024); } while (0)
#define MMA(ai, bj, At, Bt) do { __builtin_amdgcn_s_setprio(1); _Pragma("unroll") for (int m = 0; m < 4; ++m) _Pragma("unroll") for (int n = 0; n < 2; ++n) _Pragma("unroll") for (int k = 0; k < 2; ++k) \
        acc[ai][bj][m][n] = __builtin_amdgcn_mfma_i32_16x16x64_i8(At[m][k], Bt[n][k], acc[ai][bj][m][n], 0, 0, 0); __builtin_amdgcn_s_setprio(0); } while (0)
#define WAIT_V(n) asm volatile("s_waitcnt vmcnt(" #n ")" ::: "memory")
#define WAIT_L(n) asm volatile("s_waitcnt lgkmcnt(" #n ")" ::: "memory")
#define BAR __builtin_amdgcn_s_barrier()
#define SCHED __builtin_amdgcn_sched_barrier(0)

    auto wait_slice = [&](int src, int mt, int nt) {
        if (wid == 0) {
            const GAS unsigned* f = (const GAS unsigned*)flags + ((size_t)src * NCHUNK + ((lane & 1) ? (MT + nt) : mt)) * 32;
            const unsigned long long t0 = __builtin_amdgcn_s_memrealtime();
            for (;;) {
                const unsigned v = __hip_atomic_load(f, __ATOMIC_RELAXED, __HIP_MEMORY_SCOPE_SYSTEM);
                if (__all((int)(v - target) >= 0)) break;
                if (__hip_atomic_load(tmo, __ATOMIC_RELAXED, __HIP_MEMORY_SCOPE_AGENT) == call || __builtin_amdgcn_s_memrealtime() - t0 > DEADLINE_TICKS) {
                    __hip_atomic_store(tmo, call, __ATOMIC_RELAXED, __HIP_MEMORY_SCOPE_AGENT);
                    *(volatile LAS unsigned*)(lds + LDS_MISC + 4) = 1u;
                    break;
                }
                __builtin_amdgcn_s_sleep(POLL_SLEEP);
            }
            asm volatile("s_waitcnt vmcnt(0) lgkmcnt(0)" ::: "memory");
        }
        asm volatile("" ::: "memory"); BAR; asm volatile("" ::: "memory");
    };

    const int grp = (int)blockIdx.x & 7;
    const int wj = (int)blockIdx.x < NPUSH ? WPG + ((int)blockIdx.x >> 3) : (((int)blockIdx.x - NPUSH) >> 3);
    int tidx = wj;
    if ((int)blockIdx.x >= NPUSH) {
    int mt, nt;
    { const unsigned o = p.order[grp * NJOB + tidx]; mt = o & 15; nt = (o >> 4) & 63; }
    unsigned kord = nt < 16 ? p.korder[mt * 16 + nt] : p.kdef;
    i32x4 acc[2][2][4][2];
#pragma unroll
    for (int a = 0; a < 2; ++a)
#pragma unroll
        for (int b = 0; b < 2; ++b)
#pragma unroll
            for (int m = 0; m < 4; ++m)
#pragma unroll
                for (int n = 0; n < 2; ++n) acc[a][b][m][n] = (i32x4){0, 0, 0, 0};
    i32x4 At[4][2], B0[2][2], B1[2][2];

    const char* cA = XA + ((size_t)(kord & 7) * MT + mt) * CHUNKB;
    const char* cB = WB + ((size_t)(kord & 7) * NT + nt) * CHUNKB;
    wait_slice(kord & 7, mt, nt);
    int stamp_i = 1; (void)stamp_i;
    STAMP(stamp_i++);
    STAGE(SB(0, 0), cB); STAGE(SA(0, 0), cA); STAGE(SB(0, 1), cB + HTB); STAGE(SA(0, 1), cA + HTB);
    if (wr == 1) BAR;
    WAIT_V(4); BAR;
    STAGE(SB(1, 0), cB + KTB); STAGE(SA(1, 0), cA + KTB); STAGE(SB(1, 1), cB + KTB + HTB);
    WAIT_V(6); BAR;
    for (;;) {
        const int nidx = tidx + WPG;
        const bool has_next = nidx < NFULL;
        int nmt = mt, nnt = nt;
        if (has_next) { const unsigned o = p.order[grp * NJOB + nidx]; nmt = o & 15; nnt = (o >> 4) & 63; }
        const unsigned nkord = nnt < 16 ? p.korder[nmt * 16 + nnt] : p.kdef;
        for (int t = 0; t < 32; t += 2) {
            const char* a1 = cA + KTB;
            const char *a2, *b2;
            if ((t & 2) == 0) {
                a2 = cA + 2 * KTB; b2 = cB + 2 * KTB;
                if (wid == 0) {
                    const int sl = (t >> 2) + 1;
                    const int src = sl == 8 ? (nkord & 7) : ((kord >> (3 * sl)) & 7);
                    const int m2 = sl == 8 ? nmt : mt, n2 = sl == 8 ? nnt : nt;
                    const unsigned* f = flags + ((size_t)src * NCHUNK + ((lane & 1) ? (MT + n2) : m2)) * 32;
                    __builtin_amdgcn_global_load_lds(f, (LAS unsigned*)(lds + LDS_FLAG), 4, 0, 17);
                }
            }
            else {
                const int sl = (t >> 2) + 1;
                const int src = sl == 8 ? (nkord & 7) : ((kord >> (3 * sl)) & 7);
                const int m2 = sl == 8 ? nmt : mt, n2 = sl == 8 ? nnt : nt;
                a2 = XA + ((size_t)src * MT + m2) * CHUNKB;
                b2 = WB + ((size_t)src * NT + n2) * CHUNKB;
                if (sl < 8 || has_next) {
                    const unsigned f0 = *(const LAS unsigned*)(lds + LDS_FLAG), f1 = *(const LAS unsigned*)(lds + LDS_FLAG + 4);
                    const bool ok = (int)(f0 - target) >= 0 && (int)(f1 - target) >= 0;
                    if (!__builtin_amdgcn_readfirstlane((int)ok)) wait_slice(src, m2, n2);
                }
            }
            const char* a3 = a2 + KTB; const char* b3 = b2 + KTB;
            LDB(B0, 0, 0); SCHED; LDA(At, 0, 0); STAGE(SA(1, 1), a1 + HTB);
            WAIT_L(8); BAR; WAIT_L(0); MMA(0, 0, At, B0); BAR; SCHED;
            LDB(B1, 0, 1); STAGE(SB(0, 0), b2);
            BAR; WAIT_L(0); MMA(0, 1, At, B1); BAR;
            LDA(At, 0, 1); STAGE(SA(0, 0), a2);
            BAR; WAIT_L(0); MMA(1, 0, At, B0); BAR; SCHED;
            STAGE(SB(0, 1), b2 + HTB);
            WAIT_V(6); BAR; MMA(1, 1, At, B1); BAR;
            LDB(B0, 1, 0); SCHED; LDA(At, 1, 0); STAGE(SA(0, 1), a2 + HTB);
            WAIT_L(8); BAR; WAIT_L(0); MMA(0, 0, At, B0); BAR; SCHED;
            LDB(B1, 1, 1); STAGE(SB(1, 0), b3);
            BAR; WAIT_L(0); MMA(0, 1, At, B1); BAR;
            LDA(At, 1, 1); STAGE(SA(1, 0), a3);
            BAR; WAIT_L(0); MMA(1, 0, At, B0); BAR; SCHED;
            STAGE(SB(1, 1), b3 + HTB);
            WAIT_V(6); BAR; MMA(1, 1, At, B1); BAR;
            cA = a2; cB = b2;
        }
        STAMP(stamp_i++);
        {
            const int fr = lane & 15, fq = lane >> 4;
            float* obase = p.out + (size_t)(mt * 256 + wr * 64 + fq * 4) * GN + nt * 256 + wc * 64 + fr * 4;
            const float qnan = __builtin_nanf("");
            const bool bad = *(volatile LAS unsigned*)(lds + LDS_MISC + 4) != 0u;
#pragma unroll
            for (int ai = 0; ai < 2; ++ai)
#pragma unroll
                for (int m = 0; m < 4; ++m)
#pragma unroll
                    for (int j = 0; j < 4; ++j) {
                        f32x4 o;
#pragma unroll
                        for (int i = 0; i < 4; ++i) {
                            const float y = (float)acc[ai][i >> 1][m][i & 1][j] * scale;
                            const float c = __builtin_amdgcn_fmed3f(y, -60.0f, 60.0f);
                            const float e = __builtin_amdgcn_exp2f(c * -1.4426950408889634f);
                            o[i] = y * __builtin_amdgcn_rcpf(1.0f + e);
                        }
                        if (bad) o = (f32x4){qnan, qnan, qnan, qnan};
                        *(f32x4*)(obase + (size_t)(ai * 128 + m * 16 + j) * GN) = o;
                    }
        }
        STAMP(stamp_i++);
        if (!has_next) break;
#pragma unroll
        for (int a = 0; a < 2; ++a)
#pragma unroll
            for (int b = 0; b < 2; ++b)
#pragma unroll
                for (int m = 0; m < 4; ++m)
#pragma unroll
                    for (int n = 0; n < 2; ++n) acc[a][b][m][n] = (i32x4){0, 0, 0, 0};
        tidx = nidx; mt = nmt; nt = nnt; kord = nkord;
    }
    WAIT_V(0);
    if (wr == 0) BAR;
    BAR;
    }
    for (;;) {
        if (tid == 0) {
            const unsigned tk = __hip_atomic_fetch_add((GAS unsigned*)(myws + WS_QTICK + grp * 128), 1u, __ATOMIC_RELAXED, __HIP_MEMORY_SCOPE_AGENT);
            *(volatile LAS unsigned*)(lds + LDS_MISC + 8) = tk - (call - 1u) * 64u;
            asm volatile("s_waitcnt vmcnt(0) lgkmcnt(0)" ::: "memory");
        }
        asm volatile("" ::: "memory"); BAR; asm volatile("" ::: "memory");
        const unsigned qj = __builtin_amdgcn_readfirstlane(*(volatile LAS unsigned*)(lds + LDS_MISC + 8));
        asm volatile("s_waitcnt lgkmcnt(0)" ::: "memory"); BAR; asm volatile("" ::: "memory");
        if (qj >= 32u) break;
        const unsigned o = p.order[grp * NJOB + NFULL + qj];
        const int qmt = o & 15, qnt = (o >> 4) & 63, ai = (o >> 11) & 1, bj = (o >> 10) & 1;
        if (wid == 0) {
            const GAS unsigned* f = (const GAS unsigned*)flags + ((size_t)((lane >> 1) & 7) * NCHUNK + ((lane & 1) ? (MT + qnt) : qmt)) * 32;
            const unsigned long long t0 = __builtin_amdgcn_s_memrealtime();
            for (;;) {
                const unsigned v = __hip_atomic_load(f, __ATOMIC_RELAXED, __HIP_MEMORY_SCOPE_SYSTEM);
                if (__all((int)(v - target) >= 0)) break;
                if (__hip_atomic_load(tmo, __ATOMIC_RELAXED, __HIP_MEMORY_SCOPE_AGENT) == call || __builtin_amdgcn_s_memrealtime() - t0 > DEADLINE_TICKS) {
                    __hip_atomic_store(tmo, call, __ATOMIC_RELAXED, __HIP_MEMORY_SCOPE_AGENT);
                    *(volatile LAS unsigned*)(lds + LDS_MISC + 4) = 1u;
                    break;
                }
                __builtin_amdgcn_s_sleep(POLL_SLEEP);
            }
            asm volatile("s_waitcnt vmcnt(0) lgkmcnt(0)" ::: "memory");
        }
        asm volatile("" ::: "memory"); BAR; asm volatile("" ::: "memory");
        const char* qA = XA + (size_t)qmt * CHUNKB + ai * HTB;
        const char* qB = WB + (size_t)qnt * CHUNKB + bj * HTB;
#define QSTAGE(t) do { const int _t = (t) < 32 ? (t) : 31; const int _b = (t) & 3; \
            STAGE(_b * 2 * HTB, qA + (size_t)(_t >> 2) * (MT * CHUNKB) + (_t & 3) * KTB); \
            STAGE(_b * 2 * HTB + HTB, qB + (size_t)(_t >> 2) * (NT * CHUNKB) + (_t & 3) * KTB); } while (0)
        i32x4 qacc[4][2];
#pragma unroll
        for (int m = 0; m < 4; ++m)
#pragma unroll
            for (int n = 0; n < 2; ++n) qacc[m][n] = (i32x4){0, 0, 0, 0};
        QSTAGE(0); QSTAGE(1); QSTAGE(2);
        for (int t = 0; t < 32; ++t) {
            WAIT_V(8); BAR;
            QSTAGE(t + 3);
            const unsigned bo = (unsigned)(t & 3) * 2 * HTB;
            i32x4 qa[4][2], qb[2][2];
#pragma unroll
            for (int n = 0; n < 2; ++n)
#pragma unroll
                for (int k = 0; k < 2; ++k) qb[n][k] = *(const LAS i32x4*)(lds + bo + HTB + boff + n * 2048 + k * 1024);
#pragma unroll
            for (int m = 0; m < 4; ++m)
#pragma unroll
                for (int k = 0; k < 2; ++k) qa[m][k] = *(const LAS i32x4*)(lds + bo + aoff + m * 2048 + k * 1024);
#pragma unroll
            for (int k = 0; k < 2; ++k)
#pragma unroll
                for (int m = 0; m < 4; ++m)
#pragma unroll
                    for (int n = 0; n < 2; ++n) qacc[m][n] = __builtin_amdgcn_mfma_i32_16x16x64_i8(qa[m][k], qb[n][k], qacc[m][n], 0, 0, 0);
            WAIT_L(0);
        }
        WAIT_V(0);
#undef QSTAGE
        const int fr = lane & 15, fq = lane >> 4;
        float* obase = p.out + (size_t)(qmt * 256 + ai * 128 + wr * 64 + fq * 4) * GN + qnt * 256 + wc * 64 + fr * 4 + bj * 2;
        const float qnan = __builtin_nanf("");
        const bool bad = *(volatile LAS unsigned*)(lds + LDS_MISC + 4) != 0u;
#pragma unroll
        for (int m = 0; m < 4; ++m)
#pragma unroll
            for (int j = 0; j < 4; ++j) {
                float2 ov;
                float r[2];
#pragma unroll
                for (int n = 0; n < 2; ++n) {
                    const float y = (float)qacc[m][n][j] * scale;
                    const float c = __builtin_amdgcn_fmed3f(y, -60.0f, 60.0f);
                    const float e = __builtin_amdgcn_exp2f(c * -1.4426950408889634f);
                    r[n] = bad ? qnan : y * __builtin_amdgcn_rcpf(1.0f + e);
                }
                ov.x = r[0]; ov.y = r[1];
                *(float2*)(obase + (size_t)(m * 16 + j) * GN) = ov;
            }
    }
#undef SA
#undef SB
#undef STAGE
#undef LDA
#undef LDB
#undef MMA
#undef WAIT_V
#undef WAIT_L
#undef BAR
#undef SCHED
}

}

extern "C" void kernel_launch(int rank, int nranks, ncclComm_t comm,
                              hipStream_t stream,
                              void* const* d_in, const int* in_sizes, int n_in,
                              void* d_out, int out_size,
                              const int* in_dtypes, int out_dtype,
                              void* const* d_out_all,
                              void* const* d_ws_all, size_t ws_bytes) {
    (void)comm; (void)in_dtypes; (void)out_dtype; (void)d_out_all;
    if (nranks != NRANK || rank < 0 || rank >= NRANK || n_in < 4 || ws_bytes < WS_NEED ||
        in_sizes[0] != GM * KSL || in_sizes[1] != KSL * GN || out_size != GM * GN) {
        fprintf(stderr, "kernel_launch: unexpected problem shape\n");
        return;
    }
    static bool attr_set = false;
    if (!attr_set) {
        HIP_CHECK(hipFuncSetAttribute((const void*)fused_kernel, hipFuncAttributeMaxDynamicSharedMemorySize, LDS_BYTES));
        attr_set = true;
    }
    Params p;
    p.x = (const signed char*)d_in[0];
    p.w = (const signed char*)d_in[1];
    p.sx = (const float*)d_in[2];
    p.sw = (const float*)d_in[3];
    p.out = (float*)d_out;
    p.ws_all = (char* const*)d_ws_all;
    p.rank = rank;
    static unsigned short order_tab[8 * NJOB];
    static unsigned korder_tab[256];
    static bool tab_done = false;
    if (!tab_done) {
        int key[256];
        for (int mt = 0; mt < 16; ++mt)
            for (int nt = 0; nt < 16; ++nt) {
                int pos[NRANK], idx[NRANK];
                key[mt * 16 + nt] = 0;
                for (int q = 0; q < NRANK; ++q) {
                    const int px = 2 * ((mt - ROT_X * q) & 15), pw = 2 * ((nt - ROT_W * q) & 15) + 1;
                    pos[q] = px > pw ? px : pw; idx[q] = q; key[mt * 16 + nt] += pos[q];
                }
                for (int i = 1; i < NRANK; ++i)
                    for (int j = i; j > 0 && pos[idx[j]] < pos[idx[j - 1]]; --j) { const int t = idx[j]; idx[j] = idx[j - 1]; idx[j - 1] = t; }
                unsigned k = 0;
                for (int i = 0; i < NRANK; ++i) k |= (unsigned)idx[i] << (3 * i);
                korder_tab[mt * 16 + nt] = k;
            }
        for (int g = 0; g < 8; ++g) {
            const int rg = g & 3, cg = g >> 2;
            int n = 0;
            for (int kv = 0; kv <= 8 * 31; ++kv)
                for (int i = 0; i < 32; ++i) {
                    const int mt = rg + 4 * (i & 3), nt = 2 * (i >> 2) + cg;
                    if (key[mt * 16 + nt] == kv) order_tab[g * NJOB + n++] = (unsigned short)(mt | (nt << 4) | 0);
                }
            for (int i = 32; i < NFULL; ++i) order_tab[g * NJOB + n++] = (unsigned short)((rg + 4 * (i & 3)) | ((2 * (i >> 2) + cg) << 4) | 0);
            for (int i = NFULL; i < 128; ++i)
                for (int qd = 0; qd < 4; ++qd) order_tab[g * NJOB + n++] = (unsigned short)((rg + 4 * (i & 3)) | ((2 * (i >> 2) + cg) << 4) | (qd << 10));
        }
        tab_done = true;
    }
    for (int i = 0; i < 8 * NJOB; ++i) p.order[i] = order_tab[i];
    for (int i = 0; i < 256; ++i) p.korder[i] = korder_tab[i];
    p.kdef = 0;
    for (int i = 0; i < NRANK; ++i) p.kdef |= (unsigned)((rank + i) & 7) << (3 * i);
    fused_kernel<<<NBLK, 512, LDS_BYTES, stream>>>(p);
    HIP_CHECK(hipGetLastError());
}
```

```cpp
#include <hip/hip_runtime.h>
#include <rccl/rccl.h>
#include <cstdio>
#include <cstdint>

#define HIP_CHECK(expr) do { hipError_t _e = (expr); if (_e != hipSuccess) { \
    fprintf(stderr, "HIP error %s at %s:%d\n", hipGetErrorString(_e), __FILE__, __LINE__); \
    return; } } while (0)

namespace {

#define LAS __attribute__((address_space(3)))
#define GAS __attribute__((address_space(1)))
typedef int i32x4 __attribute__((ext_vector_type(4)));
typedef unsigned u32x4 __attribute__((ext_vector_type(4)));
typedef float f32x4 __attribute__((ext_vector_type(4)));

constexpr int GM = 4096, GK = 4096, GN = 16384, NRANK = 8, KSL = GK / NRANK;
constexpr int MT = GM / 256, NT = GN / 256;
constexpr unsigned KTB = 32768;
constexpr unsigned CHUNKB = 4 * KTB;
constexpr int NCHUNK = MT + NT;
constexpr int UNITS_PER_CHUNK = 32;
constexpr int NUNITS = NCHUNK * UNITS_PER_CHUNK;

constexpr size_t WS_CTR = 0;
constexpr size_t WS_TMO = 256;
constexpr size_t WS_QTICK = 1024;
constexpr size_t WS_FLAGS = 4096;
constexpr size_t WS_DATA = 1u << 20;
constexpr size_t XA_BYTES = (size_t)NRANK * MT * CHUNKB;
constexpr size_t WB_BYTES = (size_t)NRANK * NT * CHUNKB;
constexpr size_t PAR_BYTES = XA_BYTES + WB_BYTES;
constexpr size_t WS_NEED = WS_DATA + 2 * PAR_BYTES;

constexpr int NPUSH = 16;
constexpr int NBLK = 256;
constexpr int NGEMM = NBLK - NPUSH;
constexpr int NTILES = MT * NT;
constexpr int WPG = NGEMM / 8;
constexpr int NFULL = 4 * WPG;
constexpr int NJOB = NFULL + 32;

constexpr unsigned HTB = 16384;
constexpr unsigned STAGE_BYTES = 8 * HTB;
constexpr unsigned LDS_FLAG = STAGE_BYTES;
constexpr unsigned LDS_MISC = STAGE_BYTES + 256;
constexpr unsigned LDS_BYTES = STAGE_BYTES + 512;

constexpr int POLL_SLEEP = 16;
constexpr unsigned long long DEADLINE_TICKS = 50000000ull;

struct Params {
    const signed char* x;
    const signed char* w;
    const float* sx;
    const float* sw;
    float* out;
    char* const* ws_all;
    int rank;
    unsigned kdef;
    unsigned short order[8 * NJOB];
    unsigned korder[256];
};
constexpr int ROT_X = 2, ROT_W = 10;

__device__ __forceinline__ unsigned perm(unsigned hi, unsigned lo, unsigned sel) { return __builtin_amdgcn_perm(hi, lo, sel); }

__device__ __forceinline__ char* uniform_ptr(char* q) {
    const unsigned long long v = (unsigned long long)q;
    const unsigned lo = __builtin_amdgcn_readfirstlane((unsigned)v), hi = __builtin_amdgcn_readfirstlane((unsigned)(v >> 32));
    return (char*)(((unsigned long long)hi << 32) | lo);
}

struct PushUnit { unsigned a[16]; };

__device__ __forceinline__ void push_load(const Params& p, unsigned u, unsigned g, unsigned r, PushUnit& d) {
    const unsigned q = u >> 5, s = u & 31;
    if (q < 32 && !(q & 1)) {
        const unsigned mt = ((q >> 1) + ROT_X * p.rank) & 15, mf = s >> 1, kp = s & 1;
        const signed char* src = p.x + (size_t)(mt * 256 + mf * 16 + r) * KSL + kp * 256 + g * 16;
#pragma unroll
        for (int j = 0; j < 4; ++j) { const u32x4 t = *(const u32x4*)(src + j * 64); d.a[4 * j] = t[0]; d.a[4 * j + 1] = t[1]; d.a[4 * j + 2] = t[2]; d.a[4 * j + 3] = t[3]; }
    } else {
        const unsigned nt = q < 32 ? (((q >> 1) + ROT_W * p.rank) & 15) : (q - 16), wc = s & 3, kt = (s >> 2) & 3, kb = s >> 4;
        const signed char* src = p.w + (size_t)(kt * 128 + kb * 64 + g * 16) * GN + nt * 256 + wc * 64 + r * 4;
#pragma unroll
        for (int j = 0; j < 16; ++j) d.a[j] = *(const unsigned*)(src + (size_t)j * GN);
    }
}

__device__ __forceinline__ void pusher(const Params& p, unsigned call, int bid) {
    const unsigned tid = threadIdx.x, lane = tid & 63;
    const unsigned wid = __builtin_amdgcn_readfirstlane(tid >> 6);
    const unsigned gw = (unsigned)bid * 8 + wid, nw = NPUSH * 8;
    const size_t par = WS_DATA + (size_t)(call & 1) * PAR_BYTES;
    __amdgpu_buffer_rsrc_t rs[NRANK];
#pragma unroll
    for (int i = 0; i < NRANK; ++i) {
        const unsigned d = (unsigned)(p.rank + 1 + i) & 7u;
        rs[i] = __builtin_amdgcn_make_buffer_rsrc(uniform_ptr(p.ws_all[d]) + par, 0, (unsigned)PAR_BYTES, 0x00020000);
    }
    GAS unsigned* myflag = (GAS unsigned*)(p.ws_all[(unsigned)(p.rank + 1 + (lane & 7)) & 7u] + WS_FLAGS + (size_t)p.rank * NCHUNK * 128);
    const unsigned g = lane >> 4, r = lane & 15;
    PushUnit cur, nxt;
    push_load(p, gw, g, r, cur);
#pragma unroll
    for (int j = 0; j < 16; ++j) asm volatile("" :: "v"(cur.a[j]));
    unsigned prev_chunk = 0xffffffffu;
    for (unsigned u = gw; u < (unsigned)NUNITS; u += nw) {
        const unsigned q = u >> 5, s = u & 31;
        u32x4 v[4];
        unsigned doff[4], chunk;
        if (q < 32 && !(q & 1)) {
            const unsigned mt = ((q >> 1) + ROT_X * p.rank) & 15, mf = s >> 1, kp = s & 1;
            chunk = mt;
#pragma unroll
            for (int j = 0; j < 4; ++j) {
                v[j] = (u32x4){cur.a[4 * j], cur.a[4 * j + 1], cur.a[4 * j + 2], cur.a[4 * j + 3]};
                doff[j] = ((unsigned)p.rank * MT + mt) * CHUNKB + (kp * 2 + (j >> 1)) * KTB + mf * 2048 + (j & 1) * 1024 + lane * 16;
            }
        } else {
            const unsigned nt = q < 32 ? (((q >> 1) + ROT_W * p.rank) & 15) : (q - 16), wc = s & 3, kt = (s >> 2) & 3, kb = s >> 4;
            chunk = MT + nt;
#pragma unroll
            for (int b = 0; b < 4; ++b) {
                const unsigned t0 = perm(cur.a[4 * b + 1], cur.a[4 * b + 0], 0x05010400u), t1 = perm(cur.a[4 * b + 1], cur.a[4 * b + 0], 0x07030602u);
                const unsigned t2 = perm(cur.a[4 * b + 3], cur.a[4 * b + 2], 0x05010400u), t3 = perm(cur.a[4 * b + 3], cur.a[4 * b + 2], 0x07030602u);
                v[0][b] = perm(t2, t0, 0x05040100u); v[1][b] = perm(t2, t0, 0x07060302u);
                v[2][b] = perm(t3, t1, 0x05040100u); v[3][b] = perm(t3, t1, 0x07060302u);
            }
#pragma unroll
            for (int i = 0; i < 4; ++i)
                doff[i] = (unsigned)XA_BYTES + ((unsigned)p.rank * NT + nt) * CHUNKB + kt * KTB + (i >> 1) * HTB + (wc * 2 + (i & 1)) * 2048 + kb * 1024 + lane * 16;
        }
        if (u + nw < (unsigned)NUNITS) push_load(p, u + nw, g, r, nxt);
#pragma unroll
        for (int i = 0; i < NRANK; ++i)
#pragma unroll
            for (int j = 0; j < 4; ++j) __builtin_amdgcn_raw_buffer_store_b128(v[j], rs[i], doff[j], 0, 17  );
        asm volatile("s_waitcnt vmcnt(32)" ::: "memory");
        if (prev_chunk != 0xffffffffu && lane < NRANK) __hip_atomic_fetch_add(myflag + prev_chunk * 32, 1u, __ATOMIC_RELAXED, __HIP_MEMORY_SCOPE_SYSTEM);
        __builtin_amdgcn_s_barrier();
        prev_chunk = chunk;
        cur = nxt;
    }
    asm volatile("s_waitcnt vmcnt(0)" ::: "memory");
    if (lane < NRANK) __hip_atomic_fetch_add(myflag + prev_chunk * 32, 1u, __ATOMIC_RELAXED, __HIP_MEMORY_SCOPE_SYSTEM);
}

__global__ __launch_bounds__(512, 2) void fused_kernel(Params p) {
    extern __shared__ __attribute__((aligned(16))) unsigned char lds_raw[];
    LAS unsigned char* lds = (LAS unsigned char*)lds_raw;
    const int tid = threadIdx.x, lane = tid & 63;
    const int wid = __builtin_amdgcn_readfirstlane(tid >> 6);
    char* myws = uniform_ptr(p.ws_all[p.rank]);

    if (tid == 0) {
        const unsigned ticket = __hip_atomic_fetch_add((unsigned*)(myws + WS_CTR), 1u, __ATOMIC_RELAXED, __HIP_MEMORY_SCOPE_AGENT);
        *(LAS unsigned*)(lds + LDS_MISC) = ticket / gridDim.x + 1u;
        *(LAS unsigned*)(lds + LDS_MISC + 4) = 0u;
        __builtin_amdgcn_fence(__ATOMIC_ACQUIRE, "");
        asm volatile("s_waitcnt vmcnt(0) lgkmcnt(0)" ::: "memory");
    }
    __syncthreads();
    const unsigned call = __builtin_amdgcn_readfirstlane(*(LAS unsigned*)(lds + LDS_MISC));

    if ((int)blockIdx.x < NPUSH) pusher(p, call, blockIdx.x);

    const int wr = wid >> 2, wc = wid & 3;
    const unsigned target = call * UNITS_PER_CHUNK;
    const char* XA = myws + WS_DATA + (size_t)(call & 1) * PAR_BYTES;
    const char* WB = XA + XA_BYTES;
    const unsigned* flags = (const unsigned*)(myws + WS_FLAGS);
    GAS unsigned* tmo = (GAS unsigned*)(myws + WS_TMO);
    const float scale = p.sx[0] * p.sw[0];

    const unsigned voff = (unsigned)tid * 16u;
    const unsigned ldsw = (unsigned)wid * 1024u;
    const unsigned aoff = (unsigned)wr * 8192u + (unsigned)lane * 16u, boff = (unsigned)wc * 4096u + (unsigned)lane * 16u;

#define SA(b, h) (((b) * 2 + (h)) * HTB)
#define SB(b, h) ((4 + (b) * 2 + (h)) * HTB)
#define STAGE(bufoff, gbase) do { _Pragma("unroll") for (int _i = 0; _i < 2; ++_i) \
        __builtin_amdgcn_global_load_lds((const unsigned*)((const char*)(gbase) + voff + _i * 8192), (LAS unsigned*)(lds + (bufoff) + ldsw + _i * 8192), 16, 0, 0); } while (0)
#define LDA(dst, b, h) do { _Pragma("unroll") for (int m = 0; m < 4; ++m) _Pragma("unroll") for (int k = 0; k < 2; ++k) dst[m][k] = *(const LAS i32x4*)(lds + SA(b, h) + aoff + m * 2048 + k * 1024); } while (0)
#define LDB(dst, b, h) do { _Pragma("unroll") for (int n = 0; n < 2; ++n) _Pragma("unroll") for (int k = 0; k < 2; ++k) dst[n][k] = *(const LAS i32x4*)(lds + SB(b, h) + boff + n * 2048 + k * 1024); } while (0)
#define MMA(ai, bj, At, Bt) do { __builtin_amdgcn_s_setprio(1); _Pragma("unroll") for (int m = 0; m < 4; ++m) _Pragma("unroll") for (int n = 0; n < 2; ++n) _Pragma("unroll") for (int k = 0; k < 2; ++k) \
        acc[ai][bj][m][n] = __builtin_amdgcn_mfma_i32_16x16x64_i8(At[m][k], Bt[n][k], acc[ai][bj][m][n], 0, 0, 0); __builtin_amdgcn_s_setprio(0); } while (0)
#define WAIT_V(n) asm volatile("s_waitcnt vmcnt(" #n ")" ::: "memory")
#define WAIT_L(n) asm volatile("s_waitcnt lgkmcnt(" #n ")" ::: "memory")
#define BAR __builtin_amdgcn_s_barrier()
#define SCHED __builtin_amdgcn_sched_barrier(0)

    auto wait_slice = [&](int src, int mt, int nt) {
        if (wid == 0) {
            const GAS unsigned* f = (const GAS unsigned*)flags + ((size_t)src * NCHUNK + ((lane & 1) ? (MT + nt) : mt)) * 32;
            const unsigned long long t0 = __builtin_amdgcn_s_memrealtime();
            for (;;) {
                const unsigned v = __hip_atomic_load(f, __ATOMIC_RELAXED, __HIP_MEMORY_SCOPE_SYSTEM);
                if (__all((int)(v - target) >= 0)) break;
                if (__hip_atomic_load(tmo, __ATOMIC_RELAXED, __HIP_MEMORY_SCOPE_AGENT) == call || __builtin_amdgcn_s_memrealtime() - t0 > DEADLINE_TICKS) {
                    __hip_atomic_store(tmo, call, __ATOMIC_RELAXED, __HIP_MEMORY_SCOPE_AGENT);
                    *(volatile LAS unsigned*)(lds + LDS_MISC + 4) = 1u;
                    break;
                }
                __builtin_amdgcn_s_sleep(POLL_SLEEP);
            }
            asm volatile("s_waitcnt vmcnt(0) lgkmcnt(0)" ::: "memory");
        }
        asm volatile("" ::: "memory"); BAR; asm volatile("" ::: "memory");
    };

    const int grp = (int)blockIdx.x & 7;
    if ((int)blockIdx.x >= NPUSH) {
    int tidx = ((int)blockIdx.x - NPUSH) >> 3;
    int mt, nt;
    { const unsigned o = p.order[grp * NJOB + tidx]; mt = o & 15; nt = (o >> 4) & 63; }
    unsigned kord = nt < 16 ? p.korder[mt * 16 + nt] : p.kdef;
    i32x4 acc[2][2][4][2];
#pragma unroll
    for (int a = 0; a < 2; ++a)
#pragma unroll
        for (int b = 0; b < 2; ++b)
#pragma unroll
            for (int m = 0; m < 4; ++m)
#pragma unroll
                for (int n = 0; n < 2; ++n) acc[a][b][m][n] = (i32x4){0, 0, 0, 0};
    i32x4 At[4][2], B0[2][2], B1[2][2];

    const char* cA = XA + ((size_t)(kord & 7) * MT + mt) * CHUNKB;
    const char* cB = WB + ((size_t)(kord & 7) * NT + nt) * CHUNKB;
    wait_slice(kord & 7, mt, nt);
    STAGE(SB(0, 0), cB); STAGE(SA(0, 0), cA); STAGE(SB(0, 1), cB + HTB); STAGE(SA(0, 1), cA + HTB);
    if (wr == 1) BAR;
    WAIT_V(4); BAR;
    STAGE(SB(1, 0), cB + KTB); STAGE(SA(1, 0), cA + KTB); STAGE(SB(1, 1), cB + KTB + HTB);
    WAIT_V(6); BAR;
    for (;;) {
        const int nidx = tidx + WPG;
        const bool has_next = nidx < NFULL;
        int nmt = mt, nnt = nt;
        if (has_next) { const unsigned o = p.order[grp * NJOB + nidx]; nmt = o & 15; nnt = (o >> 4) & 63; }
        const unsigned nkord = nnt < 16 ? p.korder[nmt * 16 + nnt] : p.kdef;
        for (int t = 0; t < 32; t += 2) {
            const char* a1 = cA + KTB;
            const char *a2, *b2;
            if ((t & 2) == 0) {
                a2 = cA + 2 * KTB; b2 = cB + 2 * KTB;
                if (wid == 0) {
                    const int sl = (t >> 2) + 1;
                    const int src = sl == 8 ? (nkord & 7) : ((kord >> (3 * sl)) & 7);
                    const int m2 = sl == 8 ? nmt : mt, n2 = sl == 8 ? nnt : nt;
                    const unsigned* f = flags + ((size_t)src * NCHUNK + ((lane & 1) ? (MT + n2) : m2)) * 32;
                    __builtin_amdgcn_global_load_lds(f, (LAS unsigned*)(lds + LDS_FLAG), 4, 0, 17);
                }
            }
            else {
                const int sl = (t >> 2) + 1;
                const int src = sl == 8 ? (nkord & 7) : ((kord >> (3 * sl)) & 7);
                const int m2 = sl == 8 ? nmt : mt, n2 = sl == 8 ? nnt : nt;
                a2 = XA + ((size_t)src * MT + m2) * CHUNKB;
                b2 = WB + ((size_t)src * NT + n2) * CHUNKB;
                if (sl < 8 || has_next) {
                    const unsigned f0 = *(const LAS unsigned*)(lds + LDS_FLAG), f1 = *(const LAS unsigned*)(lds + LDS_FLAG + 4);
                    const bool ok = (int)(f0 - target) >= 0 && (int)(f1 - target) >= 0;
                    if (!__builtin_amdgcn_readfirstlane((int)ok)) wait_slice(src, m2, n2);
                }
            }
            const char* a3 = a2 + KTB; const char* b3 = b2 + KTB;
            LDB(B0, 0, 0); SCHED; LDA(At, 0, 0); STAGE(SA(1, 1), a1 + HTB);
            WAIT_L(8); BAR; WAIT_L(0); MMA(0, 0, At, B0); BAR; SCHED;
            LDB(B1, 0, 1); STAGE(SB(0, 0), b2);
            BAR; WAIT_L(0); MMA(0, 1, At, B1); BAR;
            LDA(At, 0, 1); STAGE(SA(0, 0), a2);
            BAR; WAIT_L(0); MMA(1, 0, At, B0); BAR; SCHED;
            STAGE(SB(0, 1), b2 + HTB);
            WAIT_V(6); BAR; MMA(1, 1, At, B1); BAR;
            LDB(B0, 1, 0); SCHED; LDA(At, 1, 0); STAGE(SA(0, 1), a2 + HTB);
            WAIT_L(8); BAR; WAIT_L(0); MMA(0, 0, At, B0); BAR; SCHED;
            LDB(B1, 1, 1); STAGE(SB(1, 0), b3);
            BAR; WAIT_L(0); MMA(0, 1, At, B1); BAR;
            LDA(At, 1, 1); STAGE(SA(1, 0), a3);
            BAR; WAIT_L(0); MMA(1, 0, At, B0); BAR; SCHED;
            STAGE(SB(1, 1), b3 + HTB);
            WAIT_V(6); BAR; MMA(1, 1, At, B1); BAR;
            cA = a2; cB = b2;
        }
        {
            const int fr = lane & 15, fq = lane >> 4;
            float* obase = p.out + (size_t)(mt * 256 + wr * 64 + fq * 4) * GN + nt * 256 + wc * 64 + fr * 4;
            const float qnan = __builtin_nanf("");
            const bool bad = *(volatile LAS unsigned*)(lds + LDS_MISC + 4) != 0u;
#pragma unroll
            for (int ai = 0; ai < 2; ++ai)
#pragma unroll
                for (int m = 0; m < 4; ++m)
#pragma unroll
                    for (int j = 0; j < 4; ++j) {
                        f32x4 o;
#pragma unroll
                        for (int i = 0; i < 4; ++i) {
                            const float y = (float)acc[ai][i >> 1][m][i & 1][j] * scale;
                            const float c = __builtin_amdgcn_fmed3f(y, -60.0f, 60.0f);
                            const float e = __builtin_amdgcn_exp2f(c * -1.4426950408889634f);
                            o[i] = y * __builtin_amdgcn_rcpf(1.0f + e);
                        }
                        if (bad) o = (f32x4){qnan, qnan, qnan, qnan};
                        *(f32x4*)(obase + (size_t)(ai * 128 + m * 16 + j) * GN) = o;
                    }
        }
        if (!has_next) break;
#pragma unroll
        for (int a = 0; a < 2; ++a)
#pragma unroll
            for (int b = 0; b < 2; ++b)
#pragma unroll
                for (int m = 0; m < 4; ++m)
#pragma unroll
                    for (int n = 0; n < 2; ++n) acc[a][b][m][n] = (i32x4){0, 0, 0, 0};
        tidx = nidx; mt = nmt; nt = nnt; kord = nkord;
    }
    WAIT_V(0);
    if (wr == 0) BAR;
    BAR;
    }
    for (;;) {
        if (tid == 0) {
            const unsigned tk = __hip_atomic_fetch_add((GAS unsigned*)(myws + WS_QTICK + grp * 128), 1u, __ATOMIC_RELAXED, __HIP_MEMORY_SCOPE_AGENT);
            *(volatile LAS unsigned*)(lds + LDS_MISC + 8) = tk - (call - 1u) * 64u;
            asm volatile("s_waitcnt vmcnt(0) lgkmcnt(0)" ::: "memory");
        }
        asm volatile("" ::: "memory"); BAR; asm volatile("" ::: "memory");
        const unsigned qj = __builtin_amdgcn_readfirstlane(*(volatile LAS unsigned*)(lds + LDS_MISC + 8));
        asm volatile("s_waitcnt lgkmcnt(0)" ::: "memory"); BAR; asm volatile("" ::: "memory");
        if (qj >= 32u) break;
        const unsigned o = p.order[grp * NJOB + NFULL + qj];
        const int qmt = o & 15, qnt = (o >> 4) & 63, ai = (o >> 11) & 1, bj = (o >> 10) & 1;
        if (wid == 0) {
            const GAS unsigned* f = (const GAS unsigned*)flags + ((size_t)((lane >> 1) & 7) * NCHUNK + ((lane & 1) ? (MT + qnt) : qmt)) * 32;
            const unsigned long long t0 = __builtin_amdgcn_s_memrealtime();
            for (;;) {
                const unsigned v = __hip_atomic_load(f, __ATOMIC_RELAXED, __HIP_MEMORY_SCOPE_SYSTEM);
                if (__all((int)(v - target) >= 0)) break;
                if (__hip_atomic_load(tmo, __ATOMIC_RELAXED, __HIP_MEMORY_SCOPE_AGENT) == call || __builtin_amdgcn_s_memrealtime() - t0 > DEADLINE_TICKS) {
                    __hip_atomic_store(tmo, call, __ATOMIC_RELAXED, __HIP_MEMORY_SCOPE_AGENT);
                    *(volatile LAS unsigned*)(lds + LDS_MISC + 4) = 1u;
                    break;
                }
                __builtin_amdgcn_s_sleep(POLL_SLEEP);
            }
            asm volatile("s_waitcnt vmcnt(0) lgkmcnt(0)" ::: "memory");
        }
        asm volatile("" ::: "memory"); BAR; asm volatile("" ::: "memory");
        const char* qA = XA + (size_t)qmt * CHUNKB + ai * HTB;
        const char* qB = WB + (size_t)qnt * CHUNKB + bj * HTB;
#define QSTAGE(t) do { const int _t = (t) < 32 ? (t) : 31; const int _b = (t) & 3; \
            STAGE(_b * 2 * HTB, qA + (size_t)(_t >> 2) * (MT * CHUNKB) + (_t & 3) * KTB); \
            STAGE(_b * 2 * HTB + HTB, qB + (size_t)(_t >> 2) * (NT * CHUNKB) + (_t & 3) * KTB); } while (0)
        i32x4 qacc[4][2];
#pragma unroll
        for (int m = 0; m < 4; ++m)
#pragma unroll
            for (int n = 0; n < 2; ++n) qacc[m][n] = (i32x4){0, 0, 0, 0};
        QSTAGE(0); QSTAGE(1); QSTAGE(2);
        for (int t = 0; t < 32; ++t) {
            WAIT_V(8); BAR;
            QSTAGE(t + 3);
            const unsigned bo = (unsigned)(t & 3) * 2 * HTB;
            i32x4 qa[4][2], qb[2][2];
#pragma unroll
            for (int n = 0; n < 2; ++n)
#pragma unroll
                for (int k = 0; k < 2; ++k) qb[n][k] = *(const LAS i32x4*)(lds + bo + HTB + boff + n * 2048 + k * 1024);
#pragma unroll
            for (int m = 0; m < 4; ++m)
#pragma unroll
                for (int k = 0; k < 2; ++k) qa[m][k] = *(const LAS i32x4*)(lds + bo + aoff + m * 2048 + k * 1024);
#pragma unroll
            for (int k = 0; k < 2; ++k)
#pragma unroll
                for (int m = 0; m < 4; ++m)
#pragma unroll
                    for (int n = 0; n < 2; ++n) qacc[m][n] = __builtin_amdgcn_mfma_i32_16x16x64_i8(qa[m][k], qb[n][k], qacc[m][n], 0, 0, 0);
            WAIT_L(0);
        }
        WAIT_V(0);
#undef QSTAGE
        const int fr = lane & 15, fq = lane >> 4;
        float* obase = p.out + (size_t)(qmt * 256 + ai * 128 + wr * 64 + fq * 4) * GN + qnt * 256 + wc * 64 + fr * 4 + bj * 2;
        const float qnan = __builtin_nanf("");
        const bool bad = *(volatile LAS unsigned*)(lds + LDS_MISC + 4) != 0u;
#pragma unroll
        for (int m = 0; m < 4; ++m)
#pragma unroll
            for (int j = 0; j < 4; ++j) {
                float2 ov;
                float r[2];
#pragma unroll
                for (int n = 0; n < 2; ++n) {
                    const float y = (float)qacc[m][n][j] * scale;
                    const float c = __builtin_amdgcn_fmed3f(y, -60.0f, 60.0f);
                    const float e = __builtin_amdgcn_exp2f(c * -1.4426950408889634f);
                    r[n] = bad ? qnan : y * __builtin_amdgcn_rcpf(1.0f + e);
                }
                ov.x = r[0]; ov.y = r[1];
                *(float2*)(obase + (size_t)(m * 16 + j) * GN) = ov;
            }
    }
#undef SA
#undef SB
#undef STAGE
#undef LDA
#undef LDB
#undef MMA
#undef WAIT_V
#undef WAIT_L
#undef BAR
#undef SCHED
}

}

extern "C" void kernel_launch(int rank, int nranks, ncclComm_t comm,
                              hipStream_t stream,
                              void* const* d_in, const int* in_sizes, int n_in,
                              void* d_out, int out_size,
                              const int* in_dtypes, int out_dtype,
                              void* const* d_out_all,
                              void* const* d_ws_all, size_t ws_bytes) {
    (void)comm; (void)in_dtypes; (void)out_dtype; (void)d_out_all;
    if (nranks != NRANK || rank < 0 || rank >= NRANK || n_in < 4 || ws_bytes < WS_NEED ||
        in_sizes[0] != GM * KSL || in_sizes[1] != KSL * GN || out_size != GM * GN) {
        fprintf(stderr, "kernel_launch: unexpected problem shape\n");
        return;
    }
    static bool attr_set = false;
    if (!attr_set) {
        HIP_CHECK(hipFuncSetAttribute((const void*)fused_kernel, hipFuncAttributeMaxDynamicSharedMemorySize, LDS_BYTES));
        attr_set = true;
    }
    Params p;
    p.x = (const signed char*)d_in[0];
    p.w = (const signed char*)d_in[1];
    p.sx = (const float*)d_in[2];
    p.sw = (const float*)d_in[3];
    p.out = (float*)d_out;
    p.ws_all = (char* const*)d_ws_all;
    p.rank = rank;
    static unsigned short order_tab[8 * NJOB];
    static unsigned korder_tab[256];
    static bool tab_done = false;
    if (!tab_done) {
        int key[256];
        for (int mt = 0; mt < 16; ++mt)
            for (int nt = 0; nt < 16; ++nt) {
                int pos[NRANK], idx[NRANK];
                key[mt * 16 + nt] = 0;
                for (int q = 0; q < NRANK; ++q) {
                    const int px = 2 * ((mt - ROT_X * q) & 15), pw = 2 * ((nt - ROT_W * q) & 15) + 1;
                    pos[q] = px > pw ? px : pw; idx[q] = q; key[mt * 16 + nt] += pos[q];
                }
                for (int i = 1; i < NRANK; ++i)
                    for (int j = i; j > 0 && pos[idx[j]] < pos[idx[j - 1]]; --j) { const int t = idx[j]; idx[j] = idx[j - 1]; idx[j - 1] = t; }
                unsigned k = 0;
                for (int i = 0; i < NRANK; ++i) k |= (unsigned)idx[i] << (3 * i);
                korder_tab[mt * 16 + nt] = k;
            }
        for (int g = 0; g < 8; ++g) {
            const int rg = g & 3, cg = g >> 2;
            int n = 0;
            for (int kv = 0; kv <= 8 * 31; ++kv)
                for (int i = 0; i < 32; ++i) {
                    const int mt = rg + 4 * (i & 3), nt = 2 * (i >> 2) + cg;
                    if (key[mt * 16 + nt] == kv) order_tab[g * NJOB + n++] = (unsigned short)(mt | (nt << 4) | 0);
                }
            for (int i = 32; i < NFULL; ++i) order_tab[g * NJOB + n++] = (unsigned short)((rg + 4 * (i & 3)) | ((2 * (i >> 2) + cg) << 4) | 0);
            for (int i = NFULL; i < 128; ++i)
                for (int qd = 0; qd < 4; ++qd) order_tab[g * NJOB + n++] = (unsigned short)((rg + 4 * (i & 3)) | ((2 * (i >> 2) + cg) << 4) | (qd << 10));
        }
        tab_done = true;
    }
    for (int i = 0; i < 8 * NJOB; ++i) p.order[i] = order_tab[i];
    for (int i = 0; i < 256; ++i) p.korder[i] = korder_tab[i];
    p.kdef = 0;
    for (int i = 0; i < NRANK; ++i) p.kdef |= (unsigned)((rank + i) & 7) << (3 * i);
    fused_kernel<<<NBLK, 512, LDS_BYTES, stream>>>(p);
    HIP_CHECK(hipGetLastError());
}
```
